```python
import numpy as np

kernel_name = "gpu_gemm_a2a_m4096_k4096_n4096_f32_none"


ABSMAX_RTOL = 0.001

COMPUTE_DTYPE = "float32"
EPILOGUE = "none"
IN_QUANT = None
OUT_QUANT = None
IN_DTYPES = {"x": "float32", "w_mat": "float32"}
WORLD_SIZE = 8

def setup_inputs():

    x = np.random.randn(4096, 4096).astype(np.float32)
    w_mat = (np.random.randn(4096, 4096) / np.sqrt(4096)).astype(np.float32)
    return {"x": x, "w_mat": w_mat}

def _epilogue(y):
    return y

def reference(x, w_mat):
    y = (x @ w_mat).astype(np.float32)
    return _epilogue(y)
```

```cpp
#include <hip/hip_runtime.h>
#include <rccl/rccl.h>
#include <cstdio>

#define HIP_CHECK(expr) do { hipError_t _e = (expr); if (_e != hipSuccess) { \
    fprintf(stderr, "HIP error %s at %s:%d\n", hipGetErrorString(_e), __FILE__, __LINE__); \
    return; } } while (0)

typedef _Float16 h8 __attribute__((ext_vector_type(8)));
typedef _Float16 h4 __attribute__((ext_vector_type(4)));
typedef float f4 __attribute__((ext_vector_type(4)));
typedef float f16v __attribute__((ext_vector_type(16)));

constexpr int NR = 8;
constexpr int MR = 512;
constexpr int KD = 4096;
constexpr int ND = 4096;
constexpr int NP = 512;
constexpr unsigned long long DEADLINE_TICKS = 20000000ull;

constexpr int W_TICKET = 0;
constexpr int W_TMO    = 32;
constexpr int W_DONE   = 64;
constexpr int W_PEERCNT = 128;
constexpr int W_FLAGS  = 1024;

struct Params {
  const float* x; const float* w; float* out_local;
  float* const* out_all; unsigned* const* ws_all;
  int rank;
};

__device__ __forceinline__ unsigned long long rt() { return __builtin_amdgcn_s_memrealtime(); }

constexpr int LDS_STRIDE = 40;

__global__ __launch_bounds__(256, 1) void fused_kernel(Params p) {
  __shared__ __attribute__((aligned(16))) _Float16 lA[2][128 * LDS_STRIDE];
  __shared__ __attribute__((aligned(16))) _Float16 lB[2][64 * LDS_STRIDE];
  __shared__ unsigned s_call;
  const int b = blockIdx.x, g = b & 7, j = b >> 3, t = threadIdx.x;
  const int ti = j & 3, tc = j >> 2;
  const int peer = (p.rank + g) & 7;
  unsigned* ws = p.ws_all[p.rank];
  if (t == 0) {
    unsigned tk = __hip_atomic_fetch_add(ws + W_TICKET, 1u, __ATOMIC_RELAXED, __HIP_MEMORY_SCOPE_AGENT);
    s_call = tk / gridDim.x + 1;
  }
  const int wv = t >> 6, lane = t & 63, wa = wv >> 1, wb = wv & 1;
  const float* xa = p.x + (size_t)(128 * ti) * KD;
  const float* wbp = p.w + peer * NP + tc * 64;
  const int arow = t >> 3, akq = t & 7;
  const int bkq = t & 7, bn4 = t >> 3;
  f4 ra[4], rb[4];
  auto gload = [&](int kt) {
#pragma unroll
    for (int u = 0; u < 4; ++u) ra[u] = *(const f4*)(xa + (size_t)(arow + 32 * u) * KD + kt * 32 + akq * 4);
    if (t < 128) {
#pragma unroll
      for (int u = 0; u < 4; ++u) rb[u] = *(const f4*)(wbp + (size_t)(kt * 32 + bkq * 4 + u) * ND + bn4 * 4);
    }
  };
  auto lstore = [&](int buf) {
#pragma unroll
    for (int u = 0; u < 4; ++u) {
      h4 h = {(_Float16)ra[u].x, (_Float16)ra[u].y, (_Float16)ra[u].z, (_Float16)ra[u].w};
      *(h4*)(&lA[buf][(arow + 32 * u) * LDS_STRIDE + akq * 4]) = h;
    }
    if (t < 128) {
      h4 h0 = {(_Float16)rb[0].x, (_Float16)rb[1].x, (_Float16)rb[2].x, (_Float16)rb[3].x};
      h4 h1 = {(_Float16)rb[0].y, (_Float16)rb[1].y, (_Float16)rb[2].y, (_Float16)rb[3].y};
      h4 h2 = {(_Float16)rb[0].z, (_Float16)rb[1].z, (_Float16)rb[2].z, (_Float16)rb[3].z};
      h4 h3 = {(_Float16)rb[0].w, (_Float16)rb[1].w, (_Float16)rb[2].w, (_Float16)rb[3].w};
      *(h4*)(&lB[buf][(bn4 * 4 + 0) * LDS_STRIDE + bkq * 4]) = h0;
      *(h4*)(&lB[buf][(bn4 * 4 + 1) * LDS_STRIDE + bkq * 4]) = h1;
      *(h4*)(&lB[buf][(bn4 * 4 + 2) * LDS_STRIDE + bkq * 4]) = h2;
      *(h4*)(&lB[buf][(bn4 * 4 + 3) * LDS_STRIDE + bkq * 4]) = h3;
    }
  };
  f16v acc0 = {}, acc1 = {};
  gload(0); lstore(0); __syncthreads();
  for (int kt = 0; kt < KD / 32; ++kt) {
    const int buf = kt & 1;
    if (kt + 1 < KD / 32) gload(kt + 1);
#pragma unroll
    for (int kk = 0; kk < 2; ++kk) {
      const int ko = kk * 16 + (lane >> 5) * 8;
      h8 a0 = *(const h8*)(&lA[buf][(wa * 64 + (lane & 31)) * LDS_STRIDE + ko]);
      h8 a1 = *(const h8*)(&lA[buf][(wa * 64 + 32 + (lane & 31)) * LDS_STRIDE + ko]);
      h8 bb = *(const h8*)(&lB[buf][(wb * 32 + (lane & 31)) * LDS_STRIDE + ko]);
      acc0 = __builtin_amdgcn_mfma_f32_32x32x16_f16(a0, bb, acc0, 0, 0, 0);
      acc1 = __builtin_amdgcn_mfma_f32_32x32x16_f16(a1, bb, acc1, 0, 0, 0);
    }
    if (kt + 1 < KD / 32) lstore(buf ^ 1);
    __syncthreads();
  }
  float* dst = p.out_all[peer] + (size_t)(p.rank * MR + 128 * ti + wa * 64) * NP + tc * 64 + wb * 32 + (lane & 31);
#pragma unroll
  for (int r = 0; r < 16; ++r) {
    const int row = (r & 3) + 8 * (r >> 2) + 4 * (lane >> 5);
    dst[(size_t)row * NP] = acc0[r];
    dst[(size_t)(row + 32) * NP] = acc1[r];
  }
  asm volatile("s_waitcnt vmcnt(0)" ::: "memory");
  __syncthreads();
  if (t == 0) {
    const unsigned call = s_call;
    __builtin_amdgcn_fence(__ATOMIC_RELEASE, "");
    asm volatile("s_waitcnt vmcnt(0)" ::: "memory");
    unsigned tk = __hip_atomic_fetch_add(ws + W_PEERCNT + 32 * g, 1u, __ATOMIC_RELEASE, __HIP_MEMORY_SCOPE_AGENT);
    if (tk == call * 32 - 1 && g != 0) {
      __builtin_amdgcn_fence(__ATOMIC_ACQUIRE, "agent");
      __builtin_amdgcn_fence(__ATOMIC_RELEASE, "");
      asm volatile("s_waitcnt vmcnt(0)" ::: "memory");
      __hip_atomic_store(p.ws_all[peer] + W_FLAGS + 32 * p.rank, call, __ATOMIC_RELAXED, __HIP_MEMORY_SCOPE_SYSTEM);
    }
    unsigned tk2 = __hip_atomic_fetch_add(ws + W_DONE, 1u, __ATOMIC_RELAXED, __HIP_MEMORY_SCOPE_AGENT);
    if (tk2 == call * gridDim.x - 1) {
      bool bad = false;
      for (int q = 0; q < NR; ++q) {
        if (q == p.rank) continue;
        unsigned long long t0 = rt();
        while (__hip_atomic_load(ws + W_FLAGS + 32 * q, __ATOMIC_RELAXED, __HIP_MEMORY_SCOPE_SYSTEM) < call) {
          if (__hip_atomic_load(ws + W_TMO, __ATOMIC_RELAXED, __HIP_MEMORY_SCOPE_AGENT) == call || rt() - t0 > DEADLINE_TICKS) {
            __hip_atomic_store(ws + W_TMO, call, __ATOMIC_RELAXED, __HIP_MEMORY_SCOPE_AGENT);
            bad = true; break;
          }
          __builtin_amdgcn_s_sleep(2);
        }
      }
      __builtin_amdgcn_fence(__ATOMIC_ACQUIRE, "");
      if (bad) p.out_local[(size_t)p.rank * MR * NP] = __builtin_nanf("");
    }
  }
}

extern "C" void kernel_launch(int rank, int nranks, ncclComm_t comm,
                              hipStream_t stream,
                              void* const* d_in, const int* in_sizes, int n_in,
                              void* d_out, int out_size,
                              const int* in_dtypes, int out_dtype,
                              void* const* d_out_all,
                              void* const* d_ws_all, size_t ws_bytes) {
  if (nranks != NR || n_in < 2 || out_size != 4096 * NP) { fprintf(stderr, "unsupported configuration\n"); return; }
  Params p;
  p.x = (const float*)d_in[0]; p.w = (const float*)d_in[1]; p.out_local = (float*)d_out;
  p.out_all = (float* const*)d_out_all; p.ws_all = (unsigned* const*)d_ws_all; p.rank = rank;
  fused_kernel<<<256, 256, 0, stream>>>(p);
}
```

```cpp
#include <hip/hip_runtime.h>
#include <rccl/rccl.h>
#include <cstdio>

#define HIP_CHECK(expr) do { hipError_t _e = (expr); if (_e != hipSuccess) { \
    fprintf(stderr, "HIP error %s at %s:%d\n", hipGetErrorString(_e), __FILE__, __LINE__); \
    return; } } while (0)

typedef _Float16 h8 __attribute__((ext_vector_type(8)));
typedef _Float16 h4 __attribute__((ext_vector_type(4)));
typedef float f4 __attribute__((ext_vector_type(4)));
typedef float f16v __attribute__((ext_vector_type(16)));
typedef unsigned v4u __attribute__((ext_vector_type(4)));
typedef unsigned long long u64;
#define GAS __attribute__((address_space(1)))
#define LAS __attribute__((address_space(3)))
typedef GAS unsigned gu32;
typedef GAS f4 gf4;
typedef volatile LAS unsigned lflag;
typedef volatile LAS u64 lflag64;
typedef volatile LAS v4u lflag128;

constexpr int NR = 8;
constexpr int MR = 512;
constexpr int KD = 4096;
constexpr int ND = 4096;
constexpr int NP = 512;
constexpr int NWG = 256;
constexpr u64 DEADLINE_TICKS = 20000000ull;

constexpr int W_TICKET  = 0;
constexpr int W_TMO     = 32;
constexpr int W_DONE    = 64;
constexpr int W_PEERCNT = 128;
constexpr int W_FLAGS   = 1024;
constexpr int W_XCC     = 2048;
constexpr int W_READY   = 4096;
constexpr int W_CONS    = 12288;
constexpr size_t SCR_OFF   = 1u << 20;
constexpr size_t SCR_BYTES = 8u << 20;
constexpr size_t WS_NEED   = SCR_OFF + SCR_BYTES;
constexpr size_t OUT_BYTES = (size_t)4096 * NP * 4;

constexpr int A_ROW    = 1040;
constexpr int B_ROW    = 144;
constexpr int B_STAGE  = 64 * B_ROW;
constexpr int LDS_A    = 0;
constexpr int NSTG     = 3;
constexpr int LDS_B    = 128 * A_ROW;
constexpr int LDS_TR   = LDS_B + NSTG * B_STAGE;
constexpr int LDS_FLAG = LDS_TR + 2048;
constexpr int LDS_TOTAL = LDS_FLAG + 64;

#define LATE_MODE 1
#define PUSH_AUX 17

#define STAMP(slot) do { } while (0)

struct Params {
  const float* x; const float* w;
  float* out_of[NR];
  unsigned* ws_of[NR];
  int rank;
};

__device__ __forceinline__ u64 rt() { return __builtin_amdgcn_s_memrealtime(); }
__device__ __forceinline__ unsigned xcc_id() { return (unsigned)__builtin_amdgcn_s_getreg(20 | (3 << 11)) & 0xfu; }
__device__ __forceinline__ unsigned rfl(unsigned v) { return __builtin_amdgcn_readfirstlane(v); }
#define RLX_AGENT __ATOMIC_RELAXED, __HIP_MEMORY_SCOPE_AGENT
#define WAIT_LGKM0() asm volatile("s_waitcnt lgkmcnt(0)" ::: "memory")
#define WAIT_VM0() asm volatile("s_waitcnt vmcnt(0)" ::: "memory")

__device__ __forceinline__ bool wait_ge(gu32* word, unsigned target, gu32* tmo, unsigned call) {
  const u64 t0 = rt();
  for (;;) {
    const unsigned v = rfl(__hip_atomic_load(word, RLX_AGENT));
    const unsigned tm = rfl(__hip_atomic_load(tmo, RLX_AGENT));
    if ((int)(v - target) >= 0) return true;
    if (tm == call || rt() - t0 > DEADLINE_TICKS) { __hip_atomic_store(tmo, call, RLX_AGENT); return false; }
    __builtin_amdgcn_s_sleep(1);
  }
}

struct Role { int g, s, i, h, grp, peer; };
#define ROLEMAP 0
__device__ __forceinline__ int blk_of(int g, int h, int i, int s) {
  return (s | (i << 2) | (h << 4)) * 8 + g;
}

__device__ __forceinline__ unsigned get_call(lflag* fl) {
  unsigned c;
  while ((c = rfl(fl[8])) == 0) __builtin_amdgcn_s_sleep(1);
  return c;
}

__device__ __forceinline__ void lds_spin_ge(lflag* w, unsigned need) {
  unsigned spins = 0; u64 t0 = 0;
  while (rfl(*w) < need) { if ((++spins & 63u) == 0) { if (t0 == 0) t0 = rt(); else if (rt() - t0 > DEADLINE_TICKS) break; } }
  asm volatile("" ::: "memory");
}

struct Frag { h8 b0, b1, a; };

template <bool FIRST>
__device__ __forceinline__ void mfma_step(char* smem, lflag* flg0, lflag* flg1, lflag* flg2, lflag* dn, unsigned abase, unsigned sb0, unsigned sb1, unsigned sb2, int lane,
                                          unsigned Tbase, bool last_step, h8 (&areg)[32], f16v& acc0, f16v& acc1, Frag& p0, Frag& p1,
                                          gu32* prev_ready, bool publish_prev, gu32* cons_word, unsigned& cons_val, GAS u64* tstamp) {
#define SBK(KK) ((KK) % 3 == 0 ? sb0 : ((KK) % 3 == 1 ? sb1 : sb2))
#define FLK(KK) ((KK) % 3 == 0 ? flg0 : ((KK) % 3 == 1 ? flg1 : flg2))
#define LDFRAG(F, KK, AK, K16, FIRST_) do { \
    F.b0 = *(const h8*)(smem + SBK(KK) + (K16) * 32); F.b1 = *(const h8*)(smem + SBK(KK) + (K16) * 32 + 32 * B_ROW); \
    if ((AK) >= 8 || FIRST_) F.a = *(const h8*)(smem + abase + (8 * ((AK) & 7) + 2 * (K16)) * 16); } while (0)
  asm volatile("" : "+s"(Tbase));
  unsigned flagv = 0;
#pragma unroll
  for (int kk = 0; kk < 16; ++kk) {
    const unsigned T = Tbase + kk;
#pragma unroll
    for (int k16 = 0; k16 < 4; ++k16) {
      Frag nx;
      if (k16 == 0) { flagv = *FLK(kk + 1); asm volatile("" ::: "memory"); }
      if (k16 < 2) {
        LDFRAG(nx, kk, kk, k16 + 2, FIRST);
        if (k16 == 1) { asm volatile("" ::: "memory"); if (lane == 0) *dn = T + 1; }
      } else if (kk < 15 || !last_step) {
        if (k16 == 2) { if (rfl(flagv) < T + 2) lds_spin_ge(FLK(kk + 1), T + 2); asm volatile("" ::: "memory"); }
        if (kk < 15) LDFRAG(nx, kk + 1, kk + 1, k16 - 2, FIRST);
        else LDFRAG(nx, 16, 0, k16 - 2, false);
      }
      if (kk == 2 && k16 == 0 && !FIRST && publish_prev) {
        WAIT_VM0(); if (lane == 0) __hip_atomic_fetch_add(prev_ready, 1u, RLX_AGENT);
      }
      if (kk == 12 && k16 == 0) cons_val = __hip_atomic_load(cons_word, RLX_AGENT);
      h8 a_;
      if (kk < 8) { if (FIRST) areg[kk * 4 + k16] = p0.a; a_ = areg[kk * 4 + k16]; } else a_ = p0.a;
      acc0 = __builtin_amdgcn_mfma_f32_32x32x16_f16(a_, p0.b0, acc0, 0, 0, 0);
      acc1 = __builtin_amdgcn_mfma_f32_32x32x16_f16(a_, p0.b1, acc1, 0, 0, 0);
      __builtin_amdgcn_sched_barrier(0);
      p0 = p1; p1 = nx;
    }
  }
#undef LDFRAG
#undef SBK
#undef FLK
}

#define TS_PTR ((GAS u64*)nullptr)
__device__ __forceinline__ void mfma_role(const Params& p, const Role& r, char* smem, lflag* fl, int v, int lane) {
  gu32* ws = (gu32*)p.ws_of[0];
  const int hh = lane >> 5, ln = lane & 31;
  const int R = 32 * v + ln;
  const unsigned abase = LDS_A + R * A_ROW + hh * 16;
  const unsigned bbase = LDS_B + ln * B_ROW + hh * 16;
  gu32* ready_word = ws + W_READY + 32 * (r.grp * 4 + v);
  gu32* cons_word = ws + W_CONS + 32 * (r.grp * 4 + v);
  h8 areg[32];
  Frag n0, n1;
  unsigned call = 0; bool same = true;
  lds_spin_ge(fl + 0, 1);
  n0.b0 = *(const h8*)(smem + bbase); n0.b1 = *(const h8*)(smem + bbase + 32 * B_ROW); n0.a = *(const h8*)(smem + abase);
  n1.b0 = *(const h8*)(smem + bbase + 32); n1.b1 = *(const h8*)(smem + bbase + 32 + 32 * B_ROW); n1.a = *(const h8*)(smem + abase + 32);
  for (int q = 0; q < 4; ++q) {
    f16v acc0 = {}, acc1 = {};
    unsigned cv = 0;
    const int s0 = q % 3, s1 = (q + 1) % 3, s2 = (q + 2) % 3;
    if (q == 0) mfma_step<true>(smem, fl + 0, fl + 1, fl + 2, fl + 4 + v, abase, bbase, bbase + B_STAGE, bbase + 2 * B_STAGE, lane, 0u, false, areg, acc0, acc1, n0, n1, ready_word, false, cons_word, cv, (GAS u64*)nullptr);
    else mfma_step<false>(smem, fl + s0, fl + s1, fl + s2, fl + 4 + v, abase, bbase + s0 * B_STAGE, bbase + s1 * B_STAGE, bbase + s2 * B_STAGE, lane, 16u * q, q == 3, areg, acc0, acc1, n0, n1, ready_word, q >= 2, cons_word, cv, TS_PTR);
    if (v == 0) STAMP(1 + q);
    if (q == 0) {
      call = get_call(fl);
      const u64 t0 = rt(); unsigned e;
      for (;;) {
        const int b2 = blk_of(r.g, r.h, r.i, lane & 3);
        e = __hip_atomic_load(ws + W_XCC + b2, RLX_AGENT);
        if (__all((e >> 4) == call)) break;
        if (rt() - t0 > DEADLINE_TICKS) { __hip_atomic_store(ws + W_TMO, call, RLX_AGENT); break; }
        __builtin_amdgcn_s_sleep(1);
      }
      same = __all((e & 15u) == (rfl(e) & 15u));
    }
    const unsigned N = (call - 1) * 4 + q;
    if ((int)(rfl(cv) - N) < 0) wait_ge(cons_word, N, ws + W_TMO, call);
    if (v == 0) STAMP(5 + q);
    {
      const __amdgpu_buffer_rsrc_t rs = __builtin_amdgcn_make_buffer_rsrc((void*)ws, 0, (int)WS_NEED, 0x00020000);
      const int soff = (int)SCR_OFF + (((r.grp * 4 + v) * 4 + r.s) << 13);
#define PSTORE(AUX) _Pragma("unroll") for (int rg = 0; rg < 4; ++rg) { \
        v4u a0 = {__float_as_uint(acc0[4 * rg]), __float_as_uint(acc0[4 * rg + 1]), __float_as_uint(acc0[4 * rg + 2]), __float_as_uint(acc0[4 * rg + 3])}; \
        v4u a1 = {__float_as_uint(acc1[4 * rg]), __float_as_uint(acc1[4 * rg + 1]), __float_as_uint(acc1[4 * rg + 2]), __float_as_uint(acc1[4 * rg + 3])}; \
        __builtin_amdgcn_raw_buffer_store_b128(a0, rs, lane * 16, soff + rg * 1024, AUX); \
        __builtin_amdgcn_raw_buffer_store_b128(a1, rs, lane * 16, soff + (4 + rg) * 1024, AUX); }
      if (same) { PSTORE(0) } else { PSTORE(17) }
#undef PSTORE
    }
    if (q == 0 || q == 3) {
      WAIT_VM0();
      if (lane == 0) __hip_atomic_fetch_add(ready_word, 1u, RLX_AGENT);
    }
    if (v == 0) STAMP(9 + q);
  }
}

__device__ __forceinline__ void loader_role(const Params& p, const Role& r, char* smem, lflag* fl, int lw, int lane) {
  gu32* ws = (gu32*)p.ws_of[0];
  const int wb_ = lane & 3, wa_ = (lane >> 2) & 3, wz_ = lane >> 4;
  const bool late = r.i >= 2;
  const __amdgpu_buffer_rsrc_t rs_w = __builtin_amdgcn_make_buffer_rsrc((void*)p.w, 0, KD * ND * 4, 0x00020000);
  const __amdgpu_buffer_rsrc_t rs_x = __builtin_amdgcn_make_buffer_rsrc((void*)p.x, 0, MR * KD * 4, 0x00020000);
  const int voffW = (((1024 * r.s + 4 * wa_) * ND) + r.peer * NP + r.h * 256 + 4 * (wb_ + 4 * wz_)) * 4;
  const int l4 = lane >> 4, ak4 = lane & 15;
  const int voffA = (((128 * r.i + l4) * KD) + 1024 * r.s + 4 * ak4) * 4;
  const int wadr = LDS_B + lw * B_STAGE + (16 * wz_ + 4 * wb_) * B_ROW + wa_ * 8;
  const int aadr = LDS_A + l4 * A_ROW + ak4 * 8;
  v4u rz[3][16];
  h4 hb[16];
  unsigned allow = NSTG;

  if (LATE_MODE == 1 && late) {
    const unsigned call = get_call(fl);
    gu32* word = ws + W_READY + 32 * ((r.grp - 2) * 4 + 0);
    const unsigned target = 4 * ((call - 1) * 4 + 1);
    const u64 t0 = rt();
    while ((int)(rfl(__hip_atomic_load(word, RLX_AGENT)) - target) < 0 && rt() - t0 < 3000) __builtin_amdgcn_s_sleep(2);
  }
#define F(x) __uint_as_float(x)
#define SB() __builtin_amdgcn_sched_barrier(0)
#define PIN_HB() asm volatile("" : "+v"(hb[0]), "+v"(hb[1]), "+v"(hb[2]), "+v"(hb[3]), "+v"(hb[4]), "+v"(hb[5]), "+v"(hb[6]), "+v"(hb[7]), "+v"(hb[8]), "+v"(hb[9]), "+v"(hb[10]), "+v"(hb[11]), "+v"(hb[12]), "+v"(hb[13]), "+v"(hb[14]), "+v"(hb[15]))
#define EXP_SPREAD 0
#define LOAD_W(buf, T) do { const int q_ = (T) >> 4, kk_ = (T) & 15; const int cq_ = ((late ? ((q_ + 2) & 3) : q_) + (EXP_SPREAD == 1 ? kk_ : (EXP_SPREAD == 2 ? r.s : (EXP_SPREAD == 3 ? r.s + kk_ : 0)))) & 3; \
    const int so_ = (64 * kk_ * ND + cq_ * 64) * 4; \
    _Pragma("unroll") for (int j = 0; j < 4; ++j) _Pragma("unroll") for (int u = 0; u < 4; ++u) \
      buf[j * 4 + u] = __builtin_amdgcn_raw_buffer_load_b128(rs_w, voffW, so_ + (16 * j + u) * ND * 4, 0); } while (0)
#define LOAD_A(buf, T, HALF) do { _Pragma("unroll") for (int u = 0; u < 16; ++u) \
      buf[u] = __builtin_amdgcn_raw_buffer_load_b128(rs_x, voffA, (4 * (16 * (HALF) + u) * KD + 64 * (T)) * 4, 0); } while (0)
#define WAIT_FREE(T) do { if ((unsigned)(T) >= allow) { unsigned spins = 0; u64 t0 = 0; for (;;) { const v4u dn = *(lflag128*)(fl + 4); \
      allow = min(min(rfl(dn.x), rfl(dn.y)), min(rfl(dn.z), rfl(dn.w))) + NSTG; if ((unsigned)(T) < allow) break; \
      if ((++spins & 63u) == 0) { if (t0 == 0) t0 = rt(); else if (rt() - t0 > DEADLINE_TICKS) break; } } asm volatile("" ::: "memory"); } } while (0)
#define CVT_W(buf) do { _Pragma("unroll") for (int j = 0; j < 4; ++j) _Pragma("unroll") for (int c = 0; c < 4; ++c) \
      hb[j * 4 + c] = (h4){(_Float16)F(buf[j * 4 + 0][c]), (_Float16)F(buf[j * 4 + 1][c]), (_Float16)F(buf[j * 4 + 2][c]), (_Float16)F(buf[j * 4 + 3][c])}; } while (0)
#define WRITE_W() do { _Pragma("unroll") for (int j = 0; j < 4; ++j) _Pragma("unroll") for (int c = 0; c < 4; ++c) \
      *(h4*)(smem + wadr + c * B_ROW + j * 32) = hb[j * 4 + c]; } while (0)
#define CVT_A(buf) do { _Pragma("unroll") for (int u = 0; u < 16; ++u) hb[u] = (h4){(_Float16)F(buf[u].x), (_Float16)F(buf[u].y), (_Float16)F(buf[u].z), (_Float16)F(buf[u].w)}; } while (0)
#define WRITE_A(T, HALF) do { const int at_ = aadr + ((T) & 7) * 128 + (HALF) * 64 * A_ROW; \
    _Pragma("unroll") for (int u = 0; u < 16; ++u) *(h4*)(smem + at_ + u * 4 * A_ROW) = hb[u]; } while (0)
#define PUBLISH(T) do { asm volatile("" ::: "memory"); if (lane == 0) fl[lw] = (T) + 1;      { if ((T) == 0) STAMP(27); if ((T) == 7) STAMP(28); if ((T) == 15) STAMP(29); if ((T) == 31) STAMP(30); if ((T) == 47) STAMP(31); if ((T) == 63) STAMP(32); } } while (0)

  if (lw == 0) STAMP(26);
  LOAD_A(rz[0], lw, 0); LOAD_A(rz[1], lw, 1); LOAD_W(rz[2], lw);
  int T = lw;
  for (; T + 3 < 16; T += 3) {
    CVT_A(rz[0]); PIN_HB(); SB(); LOAD_A(rz[0], T + 3, 0); SB();
    WAIT_FREE(T);
    WRITE_A(T, 0); SB();
    CVT_A(rz[1]); PIN_HB(); SB(); LOAD_A(rz[1], T + 3, 1); SB();
    WRITE_A(T, 1); SB();
    CVT_W(rz[2]); PIN_HB(); SB(); LOAD_W(rz[2], T + 3); SB();
    WRITE_W();
    PUBLISH(T);
  }
  CVT_A(rz[0]); PIN_HB(); SB(); LOAD_W(rz[0], T + 3); SB();
  WAIT_FREE(T);
  WRITE_A(T, 0); SB();
  CVT_A(rz[1]); PIN_HB(); SB(); LOAD_W(rz[1], T + 6); SB();
  WRITE_A(T, 1); SB();
  CVT_W(rz[2]); PIN_HB(); SB(); LOAD_W(rz[2], T + 9); SB();
  WRITE_W();
  PUBLISH(T);
  T += 3;
#define MT() 0ull
  for (int it = 0; it < 6; ++it) {
#pragma unroll
    for (int d = 0; d < 3; ++d) {
      const u64 ta = MT(); SB();
      CVT_W(rz[d]); PIN_HB(); SB();
      const u64 tb = MT(); SB();
      LOAD_W(rz[d], min(T + 9, 63)); SB();
      const u64 tc = MT(); SB();
      u64 td = tc, te = tc;
      if (it < 5 || d == 0) {
        WAIT_FREE(T);
        td = MT(); SB();
        WRITE_W();
        PUBLISH(T);
        te = MT(); SB();
      }
      T += 3;
    }
  }
#undef SB
#undef PIN_HB
#undef F
#undef LOAD_W
#undef LOAD_A
#undef WAIT_FREE
#undef WRITE_W
#undef WRITE_A
#undef CVT_W
#undef CVT_A
#undef PUBLISH
}

__device__ __forceinline__ void reducer_role(const Params& p, const Role& r, char* smem, lflag* fl, int lane) {
  gu32* ws = (gu32*)p.ws_of[0];
  const unsigned call = get_call(fl);
  const __amdgpu_buffer_rsrc_t rs_scr = __builtin_amdgcn_make_buffer_rsrc((void*)ws, 0, (int)WS_NEED, 0x00020000);
  const __amdgpu_buffer_rsrc_t rs_out = __builtin_amdgcn_make_buffer_rsrc((void*)p.out_of[r.g], 0, (int)OUT_BYTES, 0x00020000);
  float* tr = (float*)(smem + LDS_TR);
  const bool late = r.i >= 2;
  bool same = true;
  for (int q = 0; q < 4; ++q) {
    const unsigned N1 = (call - 1) * 4 + q + 1;
    bool ok = wait_ge(ws + W_READY + 32 * (r.grp * 4 + r.s), 4 * N1, ws + W_TMO, call);
    STAMP(13 + q);
    if (q == 0) {
      const u64 t0 = rt(); unsigned e;
      for (;;) {
        const int b2 = blk_of(r.g, r.h, r.i, lane & 3);
        e = __hip_atomic_load(ws + W_XCC + b2, RLX_AGENT);
        if (__all((e >> 4) == call)) break;
        if (rt() - t0 > DEADLINE_TICKS) { __hip_atomic_store(ws + W_TMO, call, RLX_AGENT); break; }
        __builtin_amdgcn_s_sleep(1);
      }
      same = __all((e & 15u) == (rfl(e) & 15u));
    }
    v4u v[4][8];
    const int cbase = (int)SCR_OFF + (((r.grp * 4 + r.s) * 4) << 13);
    if (same) {
#pragma unroll
      for (int src = 0; src < 4; ++src)
#pragma unroll
        for (int u = 0; u < 8; ++u) v[src][u] = __builtin_amdgcn_raw_buffer_load_b128(rs_scr, lane * 16, cbase + (src << 13) + u * 1024, 16);
    } else {
#pragma unroll
      for (int src = 0; src < 4; ++src)
#pragma unroll
        for (int u = 0; u < 8; ++u) v[src][u] = __builtin_amdgcn_raw_buffer_load_b128(rs_scr, lane * 16, cbase + (src << 13) + u * 1024, 17);
    }
    const unsigned tm = rfl(__hip_atomic_load(ws + W_TMO, RLX_AGENT));
    f4 sum[8];
#pragma unroll
    for (int u = 0; u < 8; ++u) {
      sum[u] = (f4){__uint_as_float(v[0][u].x), __uint_as_float(v[0][u].y), __uint_as_float(v[0][u].z), __uint_as_float(v[0][u].w)};
#pragma unroll
      for (int src = 1; src < 4; ++src) sum[u] += (f4){__uint_as_float(v[src][u].x), __uint_as_float(v[src][u].y), __uint_as_float(v[src][u].z), __uint_as_float(v[src][u].w)};
    }
    WAIT_VM0();
    if (lane == 0) __hip_atomic_fetch_add(ws + W_CONS + 32 * (r.grp * 4 + r.s), 1u, RLX_AGENT);
    STAMP(17 + q);
    if (!ok || tm == call) { const float nn = __builtin_nanf("");
#pragma unroll
      for (int u = 0; u < 8; ++u) sum[u] = (f4){nn, nn, nn, nn}; }
    const int cq = late ? ((q + 2) & 3) : q;
    const int obase = ((p.rank * MR + 128 * r.i + 32 * r.s) * NP + (4 * r.h + cq) * 64) * 4;
#pragma unroll
    for (int rg = 0; rg < 4; ++rg) {
#pragma unroll
      for (int ct = 0; ct < 2; ++ct) {
        const int col = 32 * ct + (lane & 31), row = 4 * (lane >> 5); const f4 sv = sum[ct * 4 + rg];
        tr[(row + 0) * 64 + col] = sv.x; tr[(row + 1) * 64 + col] = sv.y; tr[(row + 2) * 64 + col] = sv.z; tr[(row + 3) * 64 + col] = sv.w;
      }
      WAIT_LGKM0();
#pragma unroll
      for (int it = 0; it < 2; ++it) {
        const int idx = it * 64 + lane, row = idx >> 4, c4 = idx & 15;
        const v4u o = *(const v4u*)(tr + row * 64 + c4 * 4);
        __builtin_amdgcn_raw_buffer_store_b128(o, rs_out, obase + ((8 * rg + row) * NP + c4 * 4) * 4, 0, PUSH_AUX);
      }
      WAIT_LGKM0();
    }
    STAMP(21 + q);
  }
  WAIT_VM0();
  STAMP(25);
  if (lane == 0) {
    unsigned tk = __hip_atomic_fetch_add(ws + W_PEERCNT + 32 * r.g, 1u, __ATOMIC_RELEASE, __HIP_MEMORY_SCOPE_AGENT);
    if (tk == call * 32 - 1 && r.g != 0) {
      __builtin_amdgcn_fence(__ATOMIC_ACQUIRE, "agent");
      __builtin_amdgcn_fence(__ATOMIC_RELEASE, "");
      asm volatile("s_waitcnt vmcnt(0)" ::: "memory");
      __hip_atomic_store((gu32*)p.ws_of[r.g] + W_FLAGS + 32 * p.rank, call, __ATOMIC_RELAXED, __HIP_MEMORY_SCOPE_SYSTEM);
    }
    unsigned tk2 = __hip_atomic_fetch_add(ws + W_DONE, 1u, __ATOMIC_RELEASE, __HIP_MEMORY_SCOPE_AGENT);
    if (tk2 == call * NWG - 1) {
      __builtin_amdgcn_fence(__ATOMIC_ACQUIRE, "agent");
      bool bad = false;
      for (int pr = 0; pr < NR; ++pr) {
        if (pr == p.rank) continue;
        const u64 t0 = rt();
        while (__hip_atomic_load(ws + W_FLAGS + 32 * pr, __ATOMIC_RELAXED, __HIP_MEMORY_SCOPE_SYSTEM) < call) {
          if (__hip_atomic_load(ws + W_TMO, RLX_AGENT) == call || rt() - t0 > DEADLINE_TICKS) {
            __hip_atomic_store(ws + W_TMO, call, RLX_AGENT); bad = true; break;
          }
          __builtin_amdgcn_s_sleep(2);
        }
      }
      __builtin_amdgcn_fence(__ATOMIC_ACQUIRE, "");
      if (bad) p.out_of[0][(size_t)p.rank * MR * NP] = __builtin_nanf("");
      STAMP(33);
    }
  }
}

__global__ __launch_bounds__(512, 2) void fused_kernel(Params p) {
  __shared__ __attribute__((aligned(16))) char smem[LDS_TOTAL];
  lflag* fl = (lflag*)(smem + LDS_FLAG);
  const int b = blockIdx.x, t = threadIdx.x;
  Role r;
  r.g = b & 7; { const int j = b >> 3; r.s = j & 3; r.i = (j >> 2) & 3; r.h = j >> 4; }
  r.grp = (r.g * 2 + r.h) * 4 + r.i;
  r.peer = (p.rank + r.g) & 7;
  if (t < 16) fl[t] = 0;
  __syncthreads();
  if (t == 0) {
    gu32* ws = (gu32*)p.ws_of[0];
    const unsigned tk = __hip_atomic_fetch_add(ws + W_TICKET, 1u, RLX_AGENT);
    const unsigned call = tk / NWG + 1;
    __hip_atomic_store(ws + W_XCC + b, (call << 4) | xcc_id(), RLX_AGENT);
    fl[8] = call;
    { const int lane = 0; STAMP(0); }
  }
  const int wv = __builtin_amdgcn_readfirstlane(t >> 6), lane = t & 63;
  if (wv < 4) mfma_role(p, r, smem, fl, wv, lane);
  else if (wv < 7) loader_role(p, r, smem, fl, wv - 4, lane);
  else reducer_role(p, r, smem, fl, lane);
}

extern "C" void kernel_launch(int rank, int nranks, ncclComm_t comm,
                              hipStream_t stream,
                              void* const* d_in, const int* in_sizes, int n_in,
                              void* d_out, int out_size,
                              const int* in_dtypes, int out_dtype,
                              void* const* d_out_all,
                              void* const* d_ws_all, size_t ws_bytes) {
  if (nranks != NR || n_in < 2 || out_size != 4096 * NP || ws_bytes < WS_NEED) { fprintf(stderr, "unsupported configuration\n"); return; }
  Params p;
  p.x = (const float*)d_in[0]; p.w = (const float*)d_in[1]; p.rank = rank;
  for (int g = 0; g < NR; ++g) { p.out_of[g] = (float*)d_out_all[(rank + g) % NR]; p.ws_of[g] = (unsigned*)d_ws_all[(rank + g) % NR]; }
  fused_kernel<<<NWG, 512, 0, stream>>>(p);
}
```

```cpp
#include <hip/hip_runtime.h>
#include <rccl/rccl.h>
#include <cstdio>

#define HIP_CHECK(expr) do { hipError_t _e = (expr); if (_e != hipSuccess) { \
    fprintf(stderr, "HIP error %s at %s:%d\n", hipGetErrorString(_e), __FILE__, __LINE__); \
    return; } } while (0)

typedef _Float16 h8 __attribute__((ext_vector_type(8)));
typedef _Float16 h4 __attribute__((ext_vector_type(4)));
typedef float f4 __attribute__((ext_vector_type(4)));
typedef float f16v __attribute__((ext_vector_type(16)));
typedef unsigned v4u __attribute__((ext_vector_type(4)));
typedef unsigned long long u64;
#define GAS __attribute__((address_space(1)))
#define LAS __attribute__((address_space(3)))
typedef GAS unsigned gu32;
typedef GAS f4 gf4;
typedef volatile LAS unsigned lflag;
typedef volatile LAS u64 lflag64;
typedef volatile LAS v4u lflag128;

constexpr int NR = 8;
constexpr int MR = 512;
constexpr int KD = 4096;
constexpr int ND = 4096;
constexpr int NP = 512;
constexpr int NWG = 256;
#define SPIN_SLEEP 0
#define PRIO_MFMA 0
#define PRIO_LOAD 0
constexpr u64 DEADLINE_TICKS = 20000000ull;

constexpr int W_TICKET  = 0;
constexpr int W_TMO     = 32;
constexpr int W_DONE    = 64;
constexpr int W_PEERCNT = 128;
constexpr int W_FLAGS   = 1024;
constexpr int W_READY   = 4096;
constexpr int W_CONS    = 12288;
constexpr int W_SYNC    = 20480;
constexpr size_t SCR_OFF   = 1u << 20;
constexpr size_t SCR_BYTES = 8u << 20;
constexpr size_t WS_NEED   = SCR_OFF + SCR_BYTES;
constexpr size_t OUT_BYTES = (size_t)4096 * NP * 4;

constexpr int A_ROW    = 1040;
constexpr int B_ROW    = 144;
constexpr int B_STAGE  = 64 * B_ROW;
constexpr int LDS_A    = 0;
constexpr int NSTG     = 3;
constexpr int LDS_B    = 128 * A_ROW;
constexpr int LDS_TR   = LDS_B + NSTG * B_STAGE;
constexpr int LDS_FLAG = LDS_TR + 2048;
constexpr int LDS_TOTAL = LDS_FLAG + 64;
constexpr int FL_ABORT = 15;

#define LATE_MODE 0
#define PUSH_AUX 17

#define STAMP(slot) do { } while (0)

struct Params {
  const float* x; const float* w;
  float* out_of[NR];
  unsigned* ws_of[NR];
  int rank;
};

__device__ __forceinline__ u64 rt() { return __builtin_amdgcn_s_memrealtime(); }
__device__ __forceinline__ unsigned xcc_id() { return (unsigned)__builtin_amdgcn_s_getreg(20 | (3 << 11)) & 0xfu; }
__device__ __forceinline__ unsigned rfl(unsigned v) { return __builtin_amdgcn_readfirstlane(v); }
#define RLX_AGENT __ATOMIC_RELAXED, __HIP_MEMORY_SCOPE_AGENT
#define WAIT_LGKM0() asm volatile("s_waitcnt lgkmcnt(0)" ::: "memory")
#define WAIT_VM0() asm volatile("s_waitcnt vmcnt(0)" ::: "memory")

__device__ __forceinline__ bool wait_ge(gu32* word, unsigned target, gu32* tmo, unsigned call) {
  const u64 t0 = rt();
  for (;;) {
    const unsigned v = rfl(__hip_atomic_load(word, RLX_AGENT));
    const unsigned tm = rfl(__hip_atomic_load(tmo, RLX_AGENT));
    if ((int)(v - target) >= 0) return true;
    if (tm == call || rt() - t0 > DEADLINE_TICKS) { __hip_atomic_store(tmo, call, RLX_AGENT); return false; }
    __builtin_amdgcn_s_sleep(1);
  }
}

struct Role { int g, s, i, h, grp, peer; };
#define ROLEMAP 0
__device__ __forceinline__ int blk_of(int g, int h, int i, int s) {
  return (s | (i << 2) | (h << 4)) * 8 + g;
}

__device__ __forceinline__ unsigned get_call(lflag* fl) {
  unsigned c;
  while ((c = rfl(fl[8])) == 0) __builtin_amdgcn_s_sleep(1);
  return c;
}

__device__ __forceinline__ void lds_spin_ge(lflag* w, unsigned need, lflag* abort_word) {
  unsigned spins = 0; u64 t0 = 0;
  while (rfl(*w) < need) {
    if (SPIN_SLEEP) __builtin_amdgcn_s_sleep(SPIN_SLEEP);
    if ((++spins & 63u) == 0) {
      if (rfl(*abort_word)) break;
      if (t0 == 0) t0 = rt(); else if (rt() - t0 > DEADLINE_TICKS) { *abort_word = 1; break; }
    }
  }
  asm volatile("" ::: "memory");
}

struct Frag { h8 b0, b1, a; };

template <bool FIRST>
__device__ __forceinline__ void mfma_step(char* smem, lflag* flg0, lflag* flg1, lflag* flg2, lflag* dn, unsigned abase, unsigned sb0, unsigned sb1, unsigned sb2, int lane,
                                          unsigned Tbase, bool last_step, h8 (&areg)[32], f16v& acc0, f16v& acc1, Frag& p0, Frag& p1,
                                          gu32* prev_ready, bool publish_prev, gu32* cons_word, unsigned& cons_val, GAS u64* tstamp, lflag* abortw) {
#define SBK(KK) ((KK) % 3 == 0 ? sb0 : ((KK) % 3 == 1 ? sb1 : sb2))
#define FLK(KK) ((KK) % 3 == 0 ? flg0 : ((KK) % 3 == 1 ? flg1 : flg2))
#define LDFRAG(F, KK, AK, K16, FIRST_) do { \
    F.b0 = *(const h8*)(smem + SBK(KK) + (K16) * 32); F.b1 = *(const h8*)(smem + SBK(KK) + (K16) * 32 + 32 * B_ROW); \
    if ((AK) >= 8 || FIRST_) F.a = *(const h8*)(smem + abase + (8 * ((AK) & 7) + 2 * (K16)) * 16); } while (0)
  asm volatile("" : "+s"(Tbase));
  unsigned flagv = 0;
#pragma unroll
  for (int kk = 0; kk < 16; ++kk) {
    const unsigned T = Tbase + kk;
#pragma unroll
    for (int k16 = 0; k16 < 4; ++k16) {
      Frag nx;
      if (k16 == 0) { flagv = *FLK(kk + 1); asm volatile("" ::: "memory"); }
      if (k16 < 2) {
        LDFRAG(nx, kk, kk, k16 + 2, FIRST);
        if (k16 == 1) { asm volatile("" ::: "memory"); if (lane == 0) *dn = T + 1; }
      } else if (kk < 15 || !last_step) {
        if (k16 == 2) { if (rfl(flagv) < T + 2) lds_spin_ge(FLK(kk + 1), T + 2, abortw); asm volatile("" ::: "memory"); }
        if (kk < 15) LDFRAG(nx, kk + 1, kk + 1, k16 - 2, FIRST);
        else LDFRAG(nx, 16, 0, k16 - 2, false);
      }
      if (kk == 2 && k16 == 0 && !FIRST && publish_prev) {
        WAIT_VM0(); if (lane == 0) __hip_atomic_fetch_add(prev_ready, 1u, RLX_AGENT);
      }
      if (kk == 12 && k16 == 0) cons_val = __hip_atomic_load(cons_word, RLX_AGENT);
      h8 a_;
      if (kk < 8) { if (FIRST) areg[kk * 4 + k16] = p0.a; a_ = areg[kk * 4 + k16]; } else a_ = p0.a;
      acc0 = __builtin_amdgcn_mfma_f32_32x32x16_f16(a_, p0.b0, acc0, 0, 0, 0);
      acc1 = __builtin_amdgcn_mfma_f32_32x32x16_f16(a_, p0.b1, acc1, 0, 0, 0);
      __builtin_amdgcn_sched_barrier(0);
      p0 = p1; p1 = nx;
    }
  }
#undef LDFRAG
#undef SBK
#undef FLK
}

#define TS_PTR ((GAS u64*)nullptr)
__device__ __forceinline__ void mfma_role(const Params& p, const Role& r, char* smem, lflag* fl, int v, int lane) {
  gu32* ws = (gu32*)p.ws_of[0];
  const int hh = lane >> 5, ln = lane & 31;
  const int R = 32 * v + ln;
  const unsigned abase = LDS_A + R * A_ROW + hh * 16;
  const unsigned bbase = LDS_B + ln * B_ROW + hh * 16;
  gu32* ready_word = ws + W_READY + 32 * (r.grp * 4 + v);
  gu32* cons_word = ws + W_CONS + 32 * (r.grp * 4 + v);
  h8 areg[32];
  Frag n0, n1;
  unsigned call = 0;
  lds_spin_ge(fl + 0, 1, fl + FL_ABORT);
  n0.b0 = *(const h8*)(smem + bbase); n0.b1 = *(const h8*)(smem + bbase + 32 * B_ROW); n0.a = *(const h8*)(smem + abase);
  n1.b0 = *(const h8*)(smem + bbase + 32); n1.b1 = *(const h8*)(smem + bbase + 32 + 32 * B_ROW); n1.a = *(const h8*)(smem + abase + 32);
  for (int q = 0; q < 4; ++q) {
    f16v acc0 = {}, acc1 = {};
    unsigned cv = 0;
    const int s0 = q % 3, s1 = (q + 1) % 3, s2 = (q + 2) % 3;
    if (q == 0) mfma_step<true>(smem, fl + 0, fl + 1, fl + 2, fl + 4 + v, abase, bbase, bbase + B_STAGE, bbase + 2 * B_STAGE, lane, 0u, false, areg, acc0, acc1, n0, n1, ready_word, false, cons_word, cv, (GAS u64*)nullptr, fl + FL_ABORT);
    else mfma_step<false>(smem, fl + s0, fl + s1, fl + s2, fl + 4 + v, abase, bbase + s0 * B_STAGE, bbase + s1 * B_STAGE, bbase + s2 * B_STAGE, lane, 16u * q, q == 3, areg, acc0, acc1, n0, n1, ready_word, q >= 2, cons_word, cv, TS_PTR, fl + FL_ABORT);
    if (v == 0) STAMP(1 + q);
    if (q == 0) {
      call = get_call(fl);
    }
    const unsigned N = (call - 1) * 4 + q;
    if ((int)(rfl(cv) - N) < 0) wait_ge(cons_word, N, ws + W_TMO, call);
    if (v == 0) STAMP(5 + q);
    {
      const __amdgpu_buffer_rsrc_t rs = __builtin_amdgcn_make_buffer_rsrc((void*)ws, 0, (int)WS_NEED, 0x00020000);
      const int soff = (int)SCR_OFF + (((r.grp * 4 + v) * 4 + r.s) << 13);
#define PSTORE(AUX) _Pragma("unroll") for (int rg = 0; rg < 4; ++rg) { \
        v4u a0 = {__float_as_uint(acc0[4 * rg]), __float_as_uint(acc0[4 * rg + 1]), __float_as_uint(acc0[4 * rg + 2]), __float_as_uint(acc0[4 * rg + 3])}; \
        v4u a1 = {__float_as_uint(acc1[4 * rg]), __float_as_uint(acc1[4 * rg + 1]), __float_as_uint(acc1[4 * rg + 2]), __float_as_uint(acc1[4 * rg + 3])}; \
        __builtin_amdgcn_raw_buffer_store_b128(a0, rs, lane * 16, soff + rg * 1024, AUX); \
        __builtin_amdgcn_raw_buffer_store_b128(a1, rs, lane * 16, soff + (4 + rg) * 1024, AUX); }
      PSTORE(17)
#undef PSTORE
    }
    if (q == 0 || q == 3) {
      WAIT_VM0();
      if (lane == 0) __hip_atomic_fetch_add(ready_word, 1u, RLX_AGENT);
    }
    if (v == 0) STAMP(9 + q);
  }
}

__device__ __forceinline__ void loader_role(const Params& p, const Role& r, char* smem, lflag* fl, int lw, int lane) {
  gu32* ws = (gu32*)p.ws_of[0];
  const int wb_ = lane & 3, wa_ = (lane >> 2) & 3, wz_ = lane >> 4;
  const bool late = r.i >= 2;
  const __amdgpu_buffer_rsrc_t rs_w = __builtin_amdgcn_make_buffer_rsrc((void*)p.w, 0, KD * ND * 4, 0x00020000);
  const __amdgpu_buffer_rsrc_t rs_x = __builtin_amdgcn_make_buffer_rsrc((void*)p.x, 0, MR * KD * 4, 0x00020000);
  const int voffW = (((1024 * r.s + 4 * wa_) * ND) + r.peer * NP + r.h * 256 + 4 * (wb_ + 4 * wz_)) * 4;
  const int l4 = lane >> 4, ak4 = lane & 15;
  const int voffA = (((128 * r.i + l4) * KD) + 1024 * r.s + 4 * ak4) * 4;
  const int wadr = LDS_B + lw * B_STAGE + (16 * wz_ + 4 * wb_) * B_ROW + wa_ * 8;
  const int aadr = LDS_A + l4 * A_ROW + ak4 * 8;
  v4u rz[3][16];
  h4 hb[16];
  unsigned allow = NSTG;

  if (LATE_MODE == 1 && late) {
    const unsigned call = get_call(fl);
    gu32* word = ws + W_READY + 32 * ((r.grp - 2) * 4 + 0);
    const unsigned target = 4 * ((call - 1) * 4 + 1);
    const u64 t0 = rt();
    while ((int)(rfl(__hip_atomic_load(word, RLX_AGENT)) - target) < 0 && rt() - t0 < 3000) __builtin_amdgcn_s_sleep(2);
  }
#define F(x) __uint_as_float(x)
#define SB() __builtin_amdgcn_sched_barrier(0)
#define SYNC_STEP 0
  auto share_sync = [&](int q_) {
    const unsigned call_ = get_call(fl);
    const int nsh_ = LATE_MODE ? 2 : 4;
    gu32* w_ = ws + W_SYNC + 32 * ((((r.g * 2 + r.h) * 4 + r.s) << 1) + (LATE_MODE ? (r.i >> 1) : 0));
    if (lane == 0) __hip_atomic_fetch_add(w_, 1u, RLX_AGENT);
    const unsigned tgt_ = nsh_ * ((call_ - 1) * 4 + q_ + 1);
    const u64 t0_ = rt();
    while ((int)(rfl(__hip_atomic_load(w_, RLX_AGENT)) - tgt_) < 0 && rt() - t0_ < 1000) __builtin_amdgcn_s_sleep(1);
  };
#define PIN_HB() asm volatile("" : "+v"(hb[0]), "+v"(hb[1]), "+v"(hb[2]), "+v"(hb[3]), "+v"(hb[4]), "+v"(hb[5]), "+v"(hb[6]), "+v"(hb[7]), "+v"(hb[8]), "+v"(hb[9]), "+v"(hb[10]), "+v"(hb[11]), "+v"(hb[12]), "+v"(hb[13]), "+v"(hb[14]), "+v"(hb[15]))
#define EXP_SPREAD 0
#define LOAD_W(buf, T) do { const int q_ = (T) >> 4, kk_ = (T) & 15; const int cq_ = ((late ? ((q_ + 2) & 3) : q_) + (EXP_SPREAD == 1 ? kk_ : (EXP_SPREAD == 2 ? r.s : (EXP_SPREAD == 3 ? r.s + kk_ : 0)))) & 3; \
    const int so_ = (64 * kk_ * ND + cq_ * 64) * 4; \
    _Pragma("unroll") for (int j = 0; j < 4; ++j) _Pragma("unroll") for (int u = 0; u < 4; ++u) \
      buf[j * 4 + u] = __builtin_amdgcn_raw_buffer_load_b128(rs_w, voffW, so_ + (16 * j + u) * ND * 4, 0); } while (0)
#define LOAD_A(buf, T, HALF) do { _Pragma("unroll") for (int u = 0; u < 16; ++u) \
      buf[u] = __builtin_amdgcn_raw_buffer_load_b128(rs_x, voffA, (4 * (16 * (HALF) + u) * KD + 64 * (T)) * 4, 0); } while (0)
#define WAIT_FREE(T) do { if ((unsigned)(T) >= allow) { unsigned spins = 0; u64 t0 = 0; for (;;) { const v4u dn = *(lflag128*)(fl + 4); \
      allow = min(min(rfl(dn.x), rfl(dn.y)), min(rfl(dn.z), rfl(dn.w))) + NSTG; if ((unsigned)(T) < allow) break; \
      if ((++spins & 63u) == 0) { if (rfl(fl[FL_ABORT])) break; if (t0 == 0) t0 = rt(); else if (rt() - t0 > DEADLINE_TICKS) { fl[FL_ABORT] = 1; break; } } } asm volatile("" ::: "memory"); } } while (0)
#define CVT_W(buf) do { _Pragma("unroll") for (int j = 0; j < 4; ++j) _Pragma("unroll") for (int c = 0; c < 4; ++c) \
      hb[j * 4 + c] = (h4){(_Float16)F(buf[j * 4 + 0][c]), (_Float16)F(buf[j * 4 + 1][c]), (_Float16)F(buf[j * 4 + 2][c]), (_Float16)F(buf[j * 4 + 3][c])}; } while (0)
#define WRITE_W() do { _Pragma("unroll") for (int j = 0; j < 4; ++j) _Pragma("unroll") for (int c = 0; c < 4; ++c) \
      *(h4*)(smem + wadr + c * B_ROW + j * 32) = hb[j * 4 + c]; } while (0)
#define CVT_A(buf) do { _Pragma("unroll") for (int u = 0; u < 16; ++u) hb[u] = (h4){(_Float16)F(buf[u].x), (_Float16)F(buf[u].y), (_Float16)F(buf[u].z), (_Float16)F(buf[u].w)}; } while (0)
#define WRITE_A(T, HALF) do { const int at_ = aadr + ((T) & 7) * 128 + (HALF) * 64 * A_ROW; \
    _Pragma("unroll") for (int u = 0; u < 16; ++u) *(h4*)(smem + at_ + u * 4 * A_ROW) = hb[u]; } while (0)
#define PUBLISH(T) do { asm volatile("" ::: "memory"); if (lane == 0) fl[lw] = (T) + 1;      { if ((T) == 0) STAMP(27); if ((T) == 7) STAMP(28); if ((T) == 15) STAMP(29); if ((T) == 31) STAMP(30); if ((T) == 47) STAMP(31); if ((T) == 63) STAMP(32); } } while (0)

  if (lw == 0) STAMP(26);
  if (SYNC_STEP && lw == 0) share_sync(0);
  LOAD_A(rz[0], lw, 0); LOAD_A(rz[1], lw, 1); LOAD_W(rz[2], lw);
  int T = lw;
  for (; T + 3 < 16; T += 3) {
    CVT_A(rz[0]); PIN_HB(); SB();
    WAIT_FREE(T);
    WRITE_A(T, 0); SB();
    CVT_A(rz[1]); PIN_HB(); SB();
    WRITE_A(T, 1); SB();
    CVT_W(rz[2]); PIN_HB(); SB();
    WRITE_W();
    PUBLISH(T); SB();
    LOAD_A(rz[0], T + 3, 0); LOAD_A(rz[1], T + 3, 1); LOAD_W(rz[2], T + 3); SB();
  }
  CVT_A(rz[0]); PIN_HB(); SB();
  WAIT_FREE(T);
  WRITE_A(T, 0); SB();
  CVT_A(rz[1]); PIN_HB(); SB();
  WRITE_A(T, 1); SB();
  CVT_W(rz[2]); PIN_HB(); SB();
  WRITE_W();
  PUBLISH(T); SB();
  if (SYNC_STEP && T + 3 == 16) share_sync(1);
  LOAD_W(rz[0], T + 3); LOAD_W(rz[1], T + 6); LOAD_W(rz[2], T + 9); SB();
  T += 3;
#define MT() 0ull
  for (int it = 0; it < 6; ++it) {
#pragma unroll
    for (int d = 0; d < 3; ++d) {
      const u64 ta = MT(); SB();
      CVT_W(rz[d]); PIN_HB(); SB();
      const u64 tb = MT(); SB();
      u64 td = tb, te = tb;
      if (it < 5 || d == 0) {
        WAIT_FREE(T);
        td = MT(); SB();
        WRITE_W();
        PUBLISH(T);
        te = MT(); SB();
      }
      if (SYNC_STEP && ((T + 9) & 15) == 0 && T + 9 < 64) share_sync((T + 9) >> 4);
      LOAD_W(rz[d], min(T + 9, 63)); SB();
      const u64 tc = MT(); SB();
      T += 3;
    }
  }
#undef SB
#undef PIN_HB
#undef F
#undef LOAD_W
#undef LOAD_A
#undef WAIT_FREE
#undef WRITE_W
#undef WRITE_A
#undef CVT_W
#undef CVT_A
#undef PUBLISH
}

__device__ __forceinline__ void reducer_role(const Params& p, const Role& r, char* smem, lflag* fl, int lane) {
  gu32* ws = (gu32*)p.ws_of[0];
  const unsigned call = get_call(fl);
  const __amdgpu_buffer_rsrc_t rs_scr = __builtin_amdgcn_make_buffer_rsrc((void*)ws, 0, (int)WS_NEED, 0x00020000);
  const __amdgpu_buffer_rsrc_t rs_out = __builtin_amdgcn_make_buffer_rsrc((void*)p.out_of[r.g], 0, (int)OUT_BYTES, 0x00020000);
  float* tr = (float*)(smem + LDS_TR);
  const bool late = r.i >= 2;
  for (int q = 0; q < 4; ++q) {
    const unsigned N1 = (call - 1) * 4 + q + 1;
    bool ok = wait_ge(ws + W_READY + 32 * (r.grp * 4 + r.s), 4 * N1, ws + W_TMO, call);
    STAMP(13 + q);
    v4u v[4][8];
    const int cbase = (int)SCR_OFF + (((r.grp * 4 + r.s) * 4) << 13);
#pragma unroll
    for (int src = 0; src < 4; ++src)
#pragma unroll
      for (int u = 0; u < 8; ++u) v[src][u] = __builtin_amdgcn_raw_buffer_load_b128(rs_scr, lane * 16, cbase + (src << 13) + u * 1024, 17);
    const unsigned tm = rfl(__hip_atomic_load(ws + W_TMO, RLX_AGENT));
    f4 sum[8];
#pragma unroll
    for (int u = 0; u < 8; ++u) {
      sum[u] = (f4){__uint_as_float(v[0][u].x), __uint_as_float(v[0][u].y), __uint_as_float(v[0][u].z), __uint_as_float(v[0][u].w)};
#pragma unroll
      for (int src = 1; src < 4; ++src) sum[u] += (f4){__uint_as_float(v[src][u].x), __uint_as_float(v[src][u].y), __uint_as_float(v[src][u].z), __uint_as_float(v[src][u].w)};
    }
    WAIT_VM0();
    if (lane == 0) __hip_atomic_fetch_add(ws + W_CONS + 32 * (r.grp * 4 + r.s), 1u, RLX_AGENT);
    STAMP(17 + q);
    if (!ok || tm == call) { const float nn = __builtin_nanf("");
#pragma unroll
      for (int u = 0; u < 8; ++u) sum[u] = (f4){nn, nn, nn, nn}; }
    const int cq = late ? ((q + 2) & 3) : q;
    const int obase = ((p.rank * MR + 128 * r.i + 32 * r.s) * NP + (4 * r.h + cq) * 64) * 4;
#pragma unroll
    for (int rg = 0; rg < 4; ++rg) {
#pragma unroll
      for (int ct = 0; ct < 2; ++ct) {
        const int col = 32 * ct + (lane & 31), row = 4 * (lane >> 5); const f4 sv = sum[ct * 4 + rg];
        tr[(row + 0) * 64 + col] = sv.x; tr[(row + 1) * 64 + col] = sv.y; tr[(row + 2) * 64 + col] = sv.z; tr[(row + 3) * 64 + col] = sv.w;
      }
      WAIT_LGKM0();
#pragma unroll
      for (int it = 0; it < 2; ++it) {
        const int idx = it * 64 + lane, row = idx >> 4, c4 = idx & 15;
        const v4u o = *(const v4u*)(tr + row * 64 + c4 * 4);
        __builtin_amdgcn_raw_buffer_store_b128(o, rs_out, obase + ((8 * rg + row) * NP + c4 * 4) * 4, 0, PUSH_AUX);
      }
      WAIT_LGKM0();
    }
    STAMP(21 + q);
  }
  WAIT_VM0();
  STAMP(25);
  if (lane == 0) {
    unsigned tk = __hip_atomic_fetch_add(ws + W_PEERCNT + 32 * r.g, 1u, __ATOMIC_RELEASE, __HIP_MEMORY_SCOPE_AGENT);
    if (tk == call * 32 - 1 && r.g != 0) {
      __builtin_amdgcn_fence(__ATOMIC_ACQUIRE, "agent");
      __builtin_amdgcn_fence(__ATOMIC_RELEASE, "");
      asm volatile("s_waitcnt vmcnt(0)" ::: "memory");
      __hip_atomic_store((gu32*)p.ws_of[r.g] + W_FLAGS + 32 * p.rank, call, __ATOMIC_RELAXED, __HIP_MEMORY_SCOPE_SYSTEM);
    }
    unsigned tk2 = __hip_atomic_fetch_add(ws + W_DONE, 1u, __ATOMIC_RELEASE, __HIP_MEMORY_SCOPE_AGENT);
    if (tk2 == call * NWG - 1) {
      __builtin_amdgcn_fence(__ATOMIC_ACQUIRE, "agent");
      bool bad = false;
      for (int pr = 0; pr < NR; ++pr) {
        if (pr == p.rank) continue;
        const u64 t0 = rt();
        while (__hip_atomic_load(ws + W_FLAGS + 32 * pr, __ATOMIC_RELAXED, __HIP_MEMORY_SCOPE_SYSTEM) < call) {
          if (__hip_atomic_load(ws + W_TMO, RLX_AGENT) == call || rt() - t0 > DEADLINE_TICKS) {
            __hip_atomic_store(ws + W_TMO, call, RLX_AGENT); bad = true; break;
          }
          __builtin_amdgcn_s_sleep(2);
        }
      }
      __builtin_amdgcn_fence(__ATOMIC_ACQUIRE, "");
      if (bad) p.out_of[0][(size_t)p.rank * MR * NP] = __builtin_nanf("");
      STAMP(33);
    }
  }
}

__global__ __launch_bounds__(512, 2) void fused_kernel(Params p) {
  __shared__ __attribute__((aligned(16))) char smem[LDS_TOTAL];
  lflag* fl = (lflag*)(smem + LDS_FLAG);
  const int b = blockIdx.x, t = threadIdx.x;
  Role r;
  r.g = b & 7; { const int j = b >> 3; r.s = j & 3; r.i = (j >> 2) & 3; r.h = j >> 4; }
  r.grp = (r.g * 2 + r.h) * 4 + r.i;
  r.peer = (p.rank + r.g) & 7;
  if (t < 16) fl[t] = 0;
  __syncthreads();
  if (t == 0) {
    gu32* ws = (gu32*)p.ws_of[0];
    const unsigned tk = __hip_atomic_fetch_add(ws + W_TICKET, 1u, RLX_AGENT);
    const unsigned call = tk / NWG + 1;
    fl[8] = call;
    { const int lane = 0; STAMP(0); }
  }
  const int wv = __builtin_amdgcn_readfirstlane(t >> 6), lane = t & 63;
  if (wv < 4) { if (PRIO_MFMA) __builtin_amdgcn_s_setprio(PRIO_MFMA); mfma_role(p, r, smem, fl, wv, lane); }
  else if (wv < 7) { if (PRIO_LOAD) __builtin_amdgcn_s_setprio(PRIO_LOAD); loader_role(p, r, smem, fl, wv - 4, lane); }
  else reducer_role(p, r, smem, fl, lane);
}

extern "C" void kernel_launch(int rank, int nranks, ncclComm_t comm,
                              hipStream_t stream,
                              void* const* d_in, const int* in_sizes, int n_in,
                              void* d_out, int out_size,
                              const int* in_dtypes, int out_dtype,
                              void* const* d_out_all,
                              void* const* d_ws_all, size_t ws_bytes) {
  if (nranks != NR || n_in < 2 || out_size != 4096 * NP || ws_bytes < WS_NEED) { fprintf(stderr, "unsupported configuration\n"); return; }
  Params p;
  p.x = (const float*)d_in[0]; p.w = (const float*)d_in[1]; p.rank = rank;
  for (int g = 0; g < NR; ++g) { p.out_of[g] = (float*)d_out_all[(rank + g) % NR]; p.ws_of[g] = (unsigned*)d_ws_all[(rank + g) % NR]; }
  fused_kernel<<<NWG, 512, 0, stream>>>(p);
}
```

```cpp
#include <hip/hip_runtime.h>
#include <rccl/rccl.h>
#include <cstdio>

#define HIP_CHECK(expr) do { hipError_t _e = (expr); if (_e != hipSuccess) { \
    fprintf(stderr, "HIP error %s at %s:%d\n", hipGetErrorString(_e), __FILE__, __LINE__); \
    return; } } while (0)

typedef _Float16 h8 __attribute__((ext_vector_type(8)));
typedef _Float16 h4 __attribute__((ext_vector_type(4)));
typedef float f4 __attribute__((ext_vector_type(4)));
typedef float f16v __attribute__((ext_vector_type(16)));
typedef unsigned v4u __attribute__((ext_vector_type(4)));
typedef unsigned long long u64;
#define GAS __attribute__((address_space(1)))
#define LAS __attribute__((address_space(3)))
typedef GAS unsigned gu32;
typedef volatile LAS unsigned lflag;
typedef volatile LAS u64 lflag64;
typedef volatile LAS v4u lflag128;

constexpr int NR = 8;
constexpr int MR = 512;
constexpr int KD = 4096;
constexpr int ND = 4096;
constexpr int NP = 512;
constexpr int NWG = 256;
constexpr u64 DEADLINE_TICKS = 20000000ull;

constexpr int W_TICKET  = 0;
constexpr int W_TMO     = 32;
constexpr int W_DONE    = 64;
constexpr int W_PEERCNT = 128;
constexpr int W_FLAGS   = 1024;
constexpr int W_READY   = 4096;
constexpr int W_CONS    = 12288;
constexpr size_t SCR_OFF   = 1u << 20;
constexpr size_t SCR_BYTES = 8u << 20;
constexpr size_t WS_NEED   = SCR_OFF + SCR_BYTES;
constexpr size_t OUT_BYTES = (size_t)4096 * NP * 4;

constexpr int A_ROW    = 1040;
constexpr int B_ROW    = 144;
constexpr int B_STAGE  = 64 * B_ROW;
constexpr int LDS_A    = 0;
constexpr int NSTG     = 3;
constexpr int LDS_B    = 128 * A_ROW;
constexpr int LDS_TR   = LDS_B + NSTG * B_STAGE;
constexpr int LDS_FLAG = LDS_TR + 2048;
constexpr int LDS_TOTAL = LDS_FLAG + 64;
constexpr int FL_ABORT = 15;

constexpr int PUSH_AUX = 17;

struct Params {
  const float* x; const float* w;
  float* out_of[NR];
  unsigned* ws_of[NR];
  int rank;
};

__device__ __forceinline__ u64 rt() { return __builtin_amdgcn_s_memrealtime(); }
__device__ __forceinline__ unsigned rfl(unsigned v) { return __builtin_amdgcn_readfirstlane(v); }
#define RLX_AGENT __ATOMIC_RELAXED, __HIP_MEMORY_SCOPE_AGENT
#define WAIT_LGKM0() asm volatile("s_waitcnt lgkmcnt(0)" ::: "memory")
#define WAIT_VM0() asm volatile("s_waitcnt vmcnt(0)" ::: "memory")

__device__ __forceinline__ bool wait_ge(gu32* word, unsigned target, gu32* tmo, unsigned call) {
  const u64 t0 = rt();
  for (;;) {
    const unsigned v = rfl(__hip_atomic_load(word, RLX_AGENT));
    const unsigned tm = rfl(__hip_atomic_load(tmo, RLX_AGENT));
    if ((int)(v - target) >= 0) return true;
    if (tm == call || rt() - t0 > DEADLINE_TICKS) { __hip_atomic_store(tmo, call, RLX_AGENT); return false; }
    __builtin_amdgcn_s_sleep(1);
  }
}

struct Role { int g, s, i, h, grp, peer; };

__device__ __forceinline__ unsigned get_call(lflag* fl) {
  unsigned c;
  while ((c = rfl(fl[8])) == 0) __builtin_amdgcn_s_sleep(1);
  return c;
}

__device__ __forceinline__ void lds_spin_ge(lflag* w, unsigned need, lflag* abort_word) {
  unsigned spins = 0; u64 t0 = 0;
  while (rfl(*w) < need) {
    if ((++spins & 63u) == 0) {
      if (rfl(*abort_word)) break;
      if (t0 == 0) t0 = rt(); else if (rt() - t0 > DEADLINE_TICKS) { *abort_word = 1; break; }
    }
  }
  asm volatile("" ::: "memory");
}

struct Frag { h8 b0, b1, a; };

template <bool FIRST>
__device__ __forceinline__ void mfma_step(char* smem, lflag* flg0, lflag* flg1, lflag* flg2, lflag* dn, unsigned abase, unsigned sb0, unsigned sb1, unsigned sb2, int lane,
                                          unsigned Tbase, bool last_step, h8 (&areg)[32], f16v& acc0, f16v& acc1, Frag& p0, Frag& p1,
                                          gu32* prev_ready, bool publish_prev, gu32* cons_word, unsigned& cons_val, lflag* abortw) {
#define SBK(KK) ((KK) % 3 == 0 ? sb0 : ((KK) % 3 == 1 ? sb1 : sb2))
#define FLK(KK) ((KK) % 3 == 0 ? flg0 : ((KK) % 3 == 1 ? flg1 : flg2))
#define LDFRAG(F, KK, AK, K16, FIRST_) do { \
    F.b0 = *(const h8*)(smem + SBK(KK) + (K16) * 32); F.b1 = *(const h8*)(smem + SBK(KK) + (K16) * 32 + 32 * B_ROW); \
    if ((AK) >= 8 || FIRST_) F.a = *(const h8*)(smem + abase + (8 * ((AK) & 7) + 2 * (K16)) * 16); } while (0)
  asm volatile("" : "+s"(Tbase));
  unsigned flagv = 0;
#pragma unroll
  for (int kk = 0; kk < 16; ++kk) {
    const unsigned T = Tbase + kk;
#pragma unroll
    for (int k16 = 0; k16 < 4; ++k16) {
      Frag nx;
      if (k16 == 0) { flagv = *FLK(kk + 1); asm volatile("" ::: "memory"); }
      if (k16 < 2) {
        LDFRAG(nx, kk, kk, k16 + 2, FIRST);
        if (k16 == 1) { asm volatile("" ::: "memory"); if (lane == 0) *dn = T + 1; }
      } else if (kk < 15 || !last_step) {
        if (k16 == 2) { if (rfl(flagv) < T + 2) lds_spin_ge(FLK(kk + 1), T + 2, abortw); asm volatile("" ::: "memory"); }
        if (kk < 15) LDFRAG(nx, kk + 1, kk + 1, k16 - 2, FIRST);
        else LDFRAG(nx, 16, 0, k16 - 2, false);
      }
      if (kk == 2 && k16 == 0 && !FIRST && publish_prev) {
        WAIT_VM0(); if (lane == 0) __hip_atomic_fetch_add(prev_ready, 1u, RLX_AGENT);
      }
      if (kk == 12 && k16 == 0) cons_val = __hip_atomic_load(cons_word, RLX_AGENT);
      h8 a_;
      if (kk < 8) { if (FIRST) areg[kk * 4 + k16] = p0.a; a_ = areg[kk * 4 + k16]; } else a_ = p0.a;
      acc0 = __builtin_amdgcn_mfma_f32_32x32x16_f16(a_, p0.b0, acc0, 0, 0, 0);
      acc1 = __builtin_amdgcn_mfma_f32_32x32x16_f16(a_, p0.b1, acc1, 0, 0, 0);
      __builtin_amdgcn_sched_barrier(0);
      p0 = p1; p1 = nx;
    }
  }
#undef LDFRAG
#undef SBK
#undef FLK
}

__device__ __forceinline__ void mfma_role(const Params& p, const Role& r, char* smem, lflag* fl, int v, int lane) {
  gu32* ws = (gu32*)p.ws_of[0];
  const int hh = lane >> 5, ln = lane & 31;
  const int R = 32 * v + ln;
  const unsigned abase = LDS_A + R * A_ROW + hh * 16;
  const unsigned bbase = LDS_B + ln * B_ROW + hh * 16;
  gu32* ready_word = ws + W_READY + 32 * (r.grp * 4 + v);
  gu32* cons_word = ws + W_CONS + 32 * (r.grp * 4 + v);
  h8 areg[32];
  Frag n0, n1;
  unsigned call = 0;
  lds_spin_ge(fl + 0, 1, fl + FL_ABORT);
  n0.b0 = *(const h8*)(smem + bbase); n0.b1 = *(const h8*)(smem + bbase + 32 * B_ROW); n0.a = *(const h8*)(smem + abase);
  n1.b0 = *(const h8*)(smem + bbase + 32); n1.b1 = *(const h8*)(smem + bbase + 32 + 32 * B_ROW); n1.a = *(const h8*)(smem + abase + 32);
  for (int q = 0; q < 4; ++q) {
    f16v acc0 = {}, acc1 = {};
    unsigned cv = 0;
    const int s0 = q % 3, s1 = (q + 1) % 3, s2 = (q + 2) % 3;
    if (q == 0) mfma_step<true>(smem, fl + 0, fl + 1, fl + 2, fl + 4 + v, abase, bbase, bbase + B_STAGE, bbase + 2 * B_STAGE, lane, 0u, false, areg, acc0, acc1, n0, n1, ready_word, false, cons_word, cv, fl + FL_ABORT);
    else mfma_step<false>(smem, fl + s0, fl + s1, fl + s2, fl + 4 + v, abase, bbase + s0 * B_STAGE, bbase + s1 * B_STAGE, bbase + s2 * B_STAGE, lane, 16u * q, q == 3, areg, acc0, acc1, n0, n1, ready_word, q >= 2, cons_word, cv, fl + FL_ABORT);
    if (q == 0) {
      call = get_call(fl);
    }
    const unsigned N = (call - 1) * 4 + q;
    if ((int)(rfl(cv) - N) < 0) wait_ge(cons_word, N, ws + W_TMO, call);
    {
      const __amdgpu_buffer_rsrc_t rs = __builtin_amdgcn_make_buffer_rsrc((void*)ws, 0, (int)WS_NEED, 0x00020000);
      const int soff = (int)SCR_OFF + (((r.grp * 4 + v) * 4 + r.s) << 13);
#define PSTORE(AUX) _Pragma("unroll") for (int rg = 0; rg < 4; ++rg) { \
        v4u a0 = {__float_as_uint(acc0[4 * rg]), __float_as_uint(acc0[4 * rg + 1]), __float_as_uint(acc0[4 * rg + 2]), __float_as_uint(acc0[4 * rg + 3])}; \
        v4u a1 = {__float_as_uint(acc1[4 * rg]), __float_as_uint(acc1[4 * rg + 1]), __float_as_uint(acc1[4 * rg + 2]), __float_as_uint(acc1[4 * rg + 3])}; \
        __builtin_amdgcn_raw_buffer_store_b128(a0, rs, lane * 16, soff + rg * 1024, AUX); \
        __builtin_amdgcn_raw_buffer_store_b128(a1, rs, lane * 16, soff + (4 + rg) * 1024, AUX); }
      PSTORE(17)
#undef PSTORE
    }
    if (q == 0 || q == 3) {
      WAIT_VM0();
      if (lane == 0) __hip_atomic_fetch_add(ready_word, 1u, RLX_AGENT);
    }
  }
}

__device__ __forceinline__ void loader_role(const Params& p, const Role& r, char* smem, lflag* fl, int lw, int lane) {
  gu32* ws = (gu32*)p.ws_of[0];
  const int wb_ = lane & 3, wa_ = (lane >> 2) & 3, wz_ = lane >> 4;
  const __amdgpu_buffer_rsrc_t rs_w = __builtin_amdgcn_make_buffer_rsrc((void*)p.w, 0, KD * ND * 4, 0x00020000);
  const __amdgpu_buffer_rsrc_t rs_x = __builtin_amdgcn_make_buffer_rsrc((void*)p.x, 0, MR * KD * 4, 0x00020000);
  const int voffW = (((1024 * r.s + 4 * wa_) * ND) + r.peer * NP + r.h * 256 + 4 * (wb_ + 4 * wz_)) * 4;
  const int l4 = lane >> 4, ak4 = lane & 15;
  const int voffA = (((128 * r.i + l4) * KD) + 1024 * r.s + 4 * ak4) * 4;
  const int wadr = LDS_B + lw * B_STAGE + (16 * wz_ + 4 * wb_) * B_ROW + wa_ * 8;
  const int aadr = LDS_A + l4 * A_ROW + ak4 * 8;
  v4u rz[3][16];
  h4 hb[16];
  unsigned allow = NSTG;

#define F(x) __uint_as_float(x)
#define SB() __builtin_amdgcn_sched_barrier(0)
#define PIN_HB() asm volatile("" : "+v"(hb[0]), "+v"(hb[1]), "+v"(hb[2]), "+v"(hb[3]), "+v"(hb[4]), "+v"(hb[5]), "+v"(hb[6]), "+v"(hb[7]), "+v"(hb[8]), "+v"(hb[9]), "+v"(hb[10]), "+v"(hb[11]), "+v"(hb[12]), "+v"(hb[13]), "+v"(hb[14]), "+v"(hb[15]))
#define LOAD_W(buf, T) do { const int cq_ = (T) >> 4, kk_ = (T) & 15; \
    const int so_ = (64 * kk_ * ND + cq_ * 64) * 4; \
    _Pragma("unroll") for (int j = 0; j < 4; ++j) _Pragma("unroll") for (int u = 0; u < 4; ++u) \
      buf[j * 4 + u] = __builtin_amdgcn_raw_buffer_load_b128(rs_w, voffW, so_ + (16 * j + u) * ND * 4, 0); } while (0)
#define LOAD_A(buf, T, HALF) do { _Pragma("unroll") for (int u = 0; u < 16; ++u) \
      buf[u] = __builtin_amdgcn_raw_buffer_load_b128(rs_x, voffA, (4 * (16 * (HALF) + u) * KD + 64 * (T)) * 4, 0); } while (0)
#define WAIT_FREE(T) do { if ((unsigned)(T) >= allow) { unsigned spins = 0; u64 t0 = 0; for (;;) { const v4u dn = *(lflag128*)(fl + 4); \
      allow = min(min(rfl(dn.x), rfl(dn.y)), min(rfl(dn.z), rfl(dn.w))) + NSTG; if ((unsigned)(T) < allow) break; \
      if ((++spins & 63u) == 0) { if (rfl(fl[FL_ABORT])) break; if (t0 == 0) t0 = rt(); else if (rt() - t0 > DEADLINE_TICKS) { fl[FL_ABORT] = 1; break; } } } asm volatile("" ::: "memory"); } } while (0)
#define CVT_W(buf) do { _Pragma("unroll") for (int j = 0; j < 4; ++j) _Pragma("unroll") for (int c = 0; c < 4; ++c) \
      hb[j * 4 + c] = (h4){(_Float16)F(buf[j * 4 + 0][c]), (_Float16)F(buf[j * 4 + 1][c]), (_Float16)F(buf[j * 4 + 2][c]), (_Float16)F(buf[j * 4 + 3][c])}; } while (0)
#define WRITE_W() do { _Pragma("unroll") for (int j = 0; j < 4; ++j) _Pragma("unroll") for (int c = 0; c < 4; ++c) \
      *(h4*)(smem + wadr + c * B_ROW + j * 32) = hb[j * 4 + c]; } while (0)
#define CVT_A(buf) do { _Pragma("unroll") for (int u = 0; u < 16; ++u) hb[u] = (h4){(_Float16)F(buf[u].x), (_Float16)F(buf[u].y), (_Float16)F(buf[u].z), (_Float16)F(buf[u].w)}; } while (0)
#define WRITE_A(T, HALF) do { const int at_ = aadr + ((T) & 7) * 128 + (HALF) * 64 * A_ROW; \
    _Pragma("unroll") for (int u = 0; u < 16; ++u) *(h4*)(smem + at_ + u * 4 * A_ROW) = hb[u]; } while (0)
#define PUBLISH(T) do { asm volatile("" ::: "memory"); if (lane == 0) fl[lw] = (T) + 1;      } while (0)
  LOAD_A(rz[0], lw, 0); LOAD_A(rz[1], lw, 1); LOAD_W(rz[2], lw);
  int T = lw;
  for (; T + 3 < 16; T += 3) {
    CVT_A(rz[0]); PIN_HB(); SB();
    WAIT_FREE(T);
    WRITE_A(T, 0); SB();
    CVT_A(rz[1]); PIN_HB(); SB();
    WRITE_A(T, 1); SB();
    CVT_W(rz[2]); PIN_HB(); SB();
    WRITE_W();
    PUBLISH(T); SB();
    LOAD_A(rz[0], T + 3, 0); LOAD_A(rz[1], T + 3, 1); LOAD_W(rz[2], T + 3); SB();
  }
  CVT_A(rz[0]); PIN_HB(); SB();
  WAIT_FREE(T);
  WRITE_A(T, 0); SB();
  CVT_A(rz[1]); PIN_HB(); SB();
  WRITE_A(T, 1); SB();
  CVT_W(rz[2]); PIN_HB(); SB();
  WRITE_W();
  PUBLISH(T); SB();
  LOAD_W(rz[0], T + 3); LOAD_W(rz[1], T + 6); LOAD_W(rz[2], T + 9); SB();
  T += 3;
  for (int it = 0; it < 6; ++it) {
#pragma unroll
    for (int d = 0; d < 3; ++d) {
      CVT_W(rz[d]); PIN_HB(); SB();
      if (it < 5 || d == 0) {
        WAIT_FREE(T);
        WRITE_W();
        PUBLISH(T);
      }
      LOAD_W(rz[d], min(T + 9, 63)); SB();
      T += 3;
    }
  }
#undef SB
#undef PIN_HB
#undef F
#undef LOAD_W
#undef LOAD_A
#undef WAIT_FREE
#undef WRITE_W
#undef WRITE_A
#undef CVT_W
#undef CVT_A
#undef PUBLISH
}

__device__ __forceinline__ void reducer_role(const Params& p, const Role& r, char* smem, lflag* fl, int lane) {
  gu32* ws = (gu32*)p.ws_of[0];
  const unsigned call = get_call(fl);
  const __amdgpu_buffer_rsrc_t rs_scr = __builtin_amdgcn_make_buffer_rsrc((void*)ws, 0, (int)WS_NEED, 0x00020000);
  const __amdgpu_buffer_rsrc_t rs_out = __builtin_amdgcn_make_buffer_rsrc((void*)p.out_of[r.g], 0, (int)OUT_BYTES, 0x00020000);
  float* tr = (float*)(smem + LDS_TR);
  for (int q = 0; q < 4; ++q) {
    const unsigned N1 = (call - 1) * 4 + q + 1;
    bool ok = wait_ge(ws + W_READY + 32 * (r.grp * 4 + r.s), 4 * N1, ws + W_TMO, call);
    v4u v[4][8];
    const int cbase = (int)SCR_OFF + (((r.grp * 4 + r.s) * 4) << 13);
#pragma unroll
    for (int src = 0; src < 4; ++src)
#pragma unroll
      for (int u = 0; u < 8; ++u) v[src][u] = __builtin_amdgcn_raw_buffer_load_b128(rs_scr, lane * 16, cbase + (src << 13) + u * 1024, 17);
    const unsigned tm = rfl(__hip_atomic_load(ws + W_TMO, RLX_AGENT));
    f4 sum[8];
#pragma unroll
    for (int u = 0; u < 8; ++u) {
      sum[u] = (f4){__uint_as_float(v[0][u].x), __uint_as_float(v[0][u].y), __uint_as_float(v[0][u].z), __uint_as_float(v[0][u].w)};
#pragma unroll
      for (int src = 1; src < 4; ++src) sum[u] += (f4){__uint_as_float(v[src][u].x), __uint_as_float(v[src][u].y), __uint_as_float(v[src][u].z), __uint_as_float(v[src][u].w)};
    }
    WAIT_VM0();
    if (lane == 0) __hip_atomic_fetch_add(ws + W_CONS + 32 * (r.grp * 4 + r.s), 1u, RLX_AGENT);
    if (!ok || tm == call || rfl(fl[FL_ABORT])) { const float nn = __builtin_nanf("");
#pragma unroll
      for (int u = 0; u < 8; ++u) sum[u] = (f4){nn, nn, nn, nn}; }
    const int cq = q;
    const int obase = ((p.rank * MR + 128 * r.i + 32 * r.s) * NP + (4 * r.h + cq) * 64) * 4;
#pragma unroll
    for (int rg = 0; rg < 4; ++rg) {
#pragma unroll
      for (int ct = 0; ct < 2; ++ct) {
        const int col = 32 * ct + (lane & 31), row = 4 * (lane >> 5); const f4 sv = sum[ct * 4 + rg];
        tr[(row + 0) * 64 + col] = sv.x; tr[(row + 1) * 64 + col] = sv.y; tr[(row + 2) * 64 + col] = sv.z; tr[(row + 3) * 64 + col] = sv.w;
      }
      WAIT_LGKM0();
#pragma unroll
      for (int it = 0; it < 2; ++it) {
        const int idx = it * 64 + lane, row = idx >> 4, c4 = idx & 15;
        const v4u o = *(const v4u*)(tr + row * 64 + c4 * 4);
        __builtin_amdgcn_raw_buffer_store_b128(o, rs_out, obase + ((8 * rg + row) * NP + c4 * 4) * 4, 0, PUSH_AUX);
      }
      WAIT_LGKM0();
    }
  }
  WAIT_VM0();
  if (lane == 0) {
    unsigned tk = __hip_atomic_fetch_add(ws + W_PEERCNT + 32 * r.g, 1u, __ATOMIC_RELEASE, __HIP_MEMORY_SCOPE_AGENT);
    if (tk == call * 32 - 1 && r.g != 0) {
      __builtin_amdgcn_fence(__ATOMIC_ACQUIRE, "agent");
      __builtin_amdgcn_fence(__ATOMIC_RELEASE, "");
      asm volatile("s_waitcnt vmcnt(0)" ::: "memory");
      __hip_atomic_store((gu32*)p.ws_of[r.g] + W_FLAGS + 32 * p.rank, call, __ATOMIC_RELAXED, __HIP_MEMORY_SCOPE_SYSTEM);
    }
    unsigned tk2 = __hip_atomic_fetch_add(ws + W_DONE, 1u, __ATOMIC_RELEASE, __HIP_MEMORY_SCOPE_AGENT);
    if (tk2 == call * NWG - 1) {
      __builtin_amdgcn_fence(__ATOMIC_ACQUIRE, "agent");
      bool bad = false;
      for (int pr = 0; pr < NR; ++pr) {
        if (pr == p.rank) continue;
        const u64 t0 = rt();
        while (__hip_atomic_load(ws + W_FLAGS + 32 * pr, __ATOMIC_RELAXED, __HIP_MEMORY_SCOPE_SYSTEM) < call) {
          if (__hip_atomic_load(ws + W_TMO, RLX_AGENT) == call || rt() - t0 > DEADLINE_TICKS) {
            __hip_atomic_store(ws + W_TMO, call, RLX_AGENT); bad = true; break;
          }
          __builtin_amdgcn_s_sleep(2);
        }
      }
      __builtin_amdgcn_fence(__ATOMIC_ACQUIRE, "");
      if (bad) p.out_of[0][(size_t)p.rank * MR * NP] = __builtin_nanf("");
    }
  }
}

__global__ __launch_bounds__(512, 2) void fused_kernel(Params p) {
  __shared__ __attribute__((aligned(16))) char smem[LDS_TOTAL];
  lflag* fl = (lflag*)(smem + LDS_FLAG);
  const int b = blockIdx.x, t = threadIdx.x;
  Role r;
  r.g = b & 7; { const int j = b >> 3; r.s = j & 3; r.i = (j >> 2) & 3; r.h = j >> 4; }
  r.grp = (r.g * 2 + r.h) * 4 + r.i;
  r.peer = (p.rank + r.g) & 7;
  if (t < 16) fl[t] = 0;
  __syncthreads();
  if (t == 0) {
    gu32* ws = (gu32*)p.ws_of[0];
    const unsigned tk = __hip_atomic_fetch_add(ws + W_TICKET, 1u, RLX_AGENT);
    const unsigned call = tk / NWG + 1;
    fl[8] = call;
  }
  const int wv = __builtin_amdgcn_readfirstlane(t >> 6), lane = t & 63;
  if (wv < 4) mfma_role(p, r, smem, fl, wv, lane);
  else if (wv < 7) loader_role(p, r, smem, fl, wv - 4, lane);
  else reducer_role(p, r, smem, fl, lane);
}

extern "C" void kernel_launch(int rank, int nranks, ncclComm_t comm,
                              hipStream_t stream,
                              void* const* d_in, const int* in_sizes, int n_in,
                              void* d_out, int out_size,
                              const int* in_dtypes, int out_dtype,
                              void* const* d_out_all,
                              void* const* d_ws_all, size_t ws_bytes) {
  if (nranks != NR || n_in < 2 || out_size != 4096 * NP || ws_bytes < WS_NEED) { fprintf(stderr, "unsupported configuration\n"); return; }
  Params p;
  p.x = (const float*)d_in[0]; p.w = (const float*)d_in[1]; p.rank = rank;
  for (int g = 0; g < NR; ++g) { p.out_of[g] = (float*)d_out_all[(rank + g) % NR]; p.ws_of[g] = (unsigned*)d_ws_all[(rank + g) % NR]; }
  fused_kernel<<<NWG, 512, 0, stream>>>(p);
}
```
